```python
import numpy as np

kernel_name = "gpu_gemm_a2a_m4096_k4096_n4096_e4m3e5m2_dqsilu"


ABSMAX_RTOL = 0.05

COMPUTE_DTYPE = "float32"
EPILOGUE = "dequant_silu"
IN_QUANT = None
OUT_QUANT = None
IN_FP8 = "e4m3e5m2"
IN_DTYPES = {"x": "e4m3", "w_mat": "e5m2",
             "scale_x": "float32", "scale_w": "float32"}
WORLD_SIZE = 8

def _snap_fp8(x, mant_bits, max_abs, min_normal):
    a = np.abs(x.astype(np.float64))
    _m, _e = np.frexp(a)
    q = np.where(a >= min_normal, np.ldexp(1.0, _e - 1 - mant_bits),
                 min_normal * 2.0 ** -mant_bits)
    snapped = np.minimum(np.round(a / q) * q, max_abs)
    return (np.sign(x) * snapped).astype(np.float32)

def setup_inputs():

    x = _snap_fp8(np.random.randn(4096, 4096).astype(np.float32), 3, 448.0, 2.0 ** -6)
    w_mat = _snap_fp8(np.random.randn(4096, 4096).astype(np.float32), 2, 57344.0, 2.0 ** -14)
    scale_x = (0.02 * (1.0 + np.random.rand(1))).astype(np.float32)
    scale_w = (0.02 * (1.0 + np.random.rand(1))).astype(np.float32)
    return {"x": x, "w_mat": w_mat, "scale_x": scale_x, "scale_w": scale_w}

def reference(x, w_mat, scale_x, scale_w):
    acc = (x @ w_mat).astype(np.float32)
    y = (acc * (scale_x * scale_w)).astype(np.float32)
    return (y / (1.0 + np.exp(-np.clip(y, -60.0, 60.0)))).astype(np.float32)
```

```cpp
#include <hip/hip_runtime.h>
#include <rccl/rccl.h>
#include <cstdio>
#include <cstdint>

#define HIP_CHECK(expr) do { hipError_t _e = (expr); if (_e != hipSuccess) { \
    fprintf(stderr, "HIP error %s at %s:%d\n", hipGetErrorString(_e), __FILE__, __LINE__); \
    return; } } while (0)

namespace {

typedef int v8i __attribute__((ext_vector_type(8)));
typedef int v4i __attribute__((ext_vector_type(4)));
typedef int v2i __attribute__((ext_vector_type(2)));
typedef float f4 __attribute__((ext_vector_type(4)));
typedef __attribute__((address_space(3))) v2i* lds_v2i_ptr;

constexpr int NRANK = 8;
constexpr int MLOC = 512;
constexpr int KDIM = 4096;
constexpr int NDIM = 4096;
constexpr int NBLK = 512;
constexpr int NWG = 256;
constexpr int WG_PER_DEST = NWG / NRANK;
constexpr int KSTEP = 128;
constexpr int KSTEPS = KDIM / KSTEP;

#define TILE_M 64
#define TILE_N 128
#define NSTAGE 6
#define NWAVE_M 2
#define NWAVE_N 4
#define NWAVE_L 6
constexpr int NWC = NWAVE_M * NWAVE_N;
constexpr int NWL = NWAVE_L;
constexpr int NTHREADS = (NWC + NWL + 1) * 64;

constexpr size_t WS_DONE = 0;
constexpr size_t WS_TMO = 128;
constexpr size_t WS_FLAG0 = 256;
constexpr size_t WS_FLAG_STRIDE = 128;

struct Params {
  const unsigned char* x;
  const unsigned char* w;
  const float* scale_x;
  const float* scale_w;
  float* out_all[NRANK];
  unsigned char* ws_all[NRANK];
  int rank;
};

__device__ __forceinline__ int swz_x(int row) { return ((row >> 1) & 1) | (row & 4); }
template <int TN>
__device__ __forceinline__ int swz_w(int k) {
  if (TN == 64) return ((k >> 2) & 1) | (((k >> 5) & 1) << 1);
  if (TN == 128) return ((k >> 1) & 1) | (((k >> 2) & 1) << 1) | (((k >> 5) & 1) << 2);
  return ((k >> 5) & 1);
}

template <int N>
__device__ __forceinline__ void wait_vm() { asm volatile("s_waitcnt vmcnt(%0)" :: "n"(N) : "memory"); }
__device__ __forceinline__ void wait_lgkm0() { asm volatile("s_waitcnt lgkmcnt(0)" ::: "memory"); }

__device__ __forceinline__ void bar() { asm volatile("s_barrier" ::: "memory"); }

__device__ __forceinline__ void store16_wt(float* p, f4 v) {
  asm volatile("global_store_dwordx4 %0, %1, off sc0 sc1\n s_nop 1" :: "v"(p), "v"(v) : "memory");
}

template <int TM, int TN, int NST>
__global__ void __launch_bounds__(NTHREADS, 1) gemm_a2a_kernel(Params P) {
  constexpr int TILES_N = NBLK / TN;
  constexpr int TILES_PER_DEST = (MLOC / TM) * TILES_N;
  constexpr int ROUNDS = TILES_PER_DEST / WG_PER_DEST;
  constexpr int TOTAL = ROUNDS * KSTEPS;
  constexpr int XBYTES = TM * KSTEP;
  constexpr int WBYTES = KSTEP * TN;
  constexpr int STAGE = XBYTES + WBYTES;
  constexpr int XP = TM / 8;
  constexpr int WP = TN / 8;
  constexpr int PER = (XP + WP) / NWL;
  constexpr int WM = TM / NWAVE_M, WN = TN / NWAVE_N;
  constexpr int MI = WM / 16, NI = WN / 16;
  constexpr int SPITCH = TN * 4 + 16;
  constexpr bool ALIAS_STG = (ROUNDS == 1);
  constexpr int STG_OFF = ALIAS_STG ? 0 : NST * STAGE;
  static_assert((XP + WP) % NWL == 0, "pieces must split over the loader waves");
  static_assert(PER * (NST - 1) < 60, "vmcnt range");

  extern __shared__ __attribute__((aligned(1024))) unsigned char smem[];

  const int tid = threadIdx.x;
  const int lane = tid & 63;
  const int wv = tid >> 6;
  const int wg = blockIdx.x;
  const int dest = wg % NRANK;
  const int idx = wg / NRANK;
  const int rank = P.rank;

  if (wv < NWC) {
    const int wm = wv / NWAVE_N, wn = wv % NWAVE_N;
    const float scale = P.scale_x[0] * P.scale_w[0];
    f4 acc[MI][NI];
#pragma unroll
    for (int a = 0; a < MI; ++a)
#pragma unroll
      for (int b = 0; b < NI; ++b) acc[a][b] = f4{0.f, 0.f, 0.f, 0.f};

    unsigned xoff[MI][2];
#pragma unroll
    for (int mi = 0; mi < MI; ++mi)
#pragma unroll
      for (int h = 0; h < 2; ++h) {
        const int row = wm * WM + mi * 16 + (lane & 15);
        const int c = 2 * (lane >> 4) + h;
        xoff[mi][h] = row * KSTEP + ((c ^ swz_x(row)) * 16);
      }
    unsigned woff[NI][4];
#pragma unroll
    for (int ni = 0; ni < NI; ++ni)
#pragma unroll
      for (int q = 0; q < 4; ++q) {
        const int g = lane >> 4, j = lane & 15;
        const int krow = 32 * g + 8 * q + (j >> 1);
        const int nt = (wn * WN + ni * 16) / 16;
        woff[ni][q] = XBYTES + krow * TN + ((nt ^ swz_w<TN>(krow)) * 16) + 8 * (j & 1);
      }

    v4i xf[2][MI][2];
    v2i wf[2][NI][4];

    auto read_frags = [&](int set, int step) {
      const unsigned sb = (unsigned)((step % NST) * STAGE);
#pragma unroll
      for (int ni = 0; ni < NI; ++ni)
#pragma unroll
        for (int q = 0; q < 4; ++q)
          asm volatile("ds_read_b64_tr_b8 %0, %1" : "=v"(wf[set][ni][q]) : "v"(woff[ni][q] + sb) : "memory");
#pragma unroll
      for (int mi = 0; mi < MI; ++mi)
#pragma unroll
        for (int h = 0; h < 2; ++h)
          asm volatile("ds_read_b128 %0, %1" : "=v"(xf[set][mi][h]) : "v"(xoff[mi][h] + sb) : "memory");
    };
    auto tie_frags = [&](int set) {
      asm volatile("s_waitcnt lgkmcnt(0)" ::: "memory");
#pragma unroll
      for (int ni = 0; ni < NI; ++ni)
#pragma unroll
        for (int q = 0; q < 4; ++q) asm volatile("" : "+v"(wf[set][ni][q]));
#pragma unroll
      for (int mi = 0; mi < MI; ++mi)
#pragma unroll
        for (int h = 0; h < 2; ++h) asm volatile("" : "+v"(xf[set][mi][h]));
    };
    auto mma = [&](int set) {
#pragma unroll
      for (int mi = 0; mi < MI; ++mi)
#pragma unroll
        for (int ni = 0; ni < NI; ++ni) {
          const v8i a = v8i{wf[set][ni][0][0], wf[set][ni][0][1], wf[set][ni][1][0], wf[set][ni][1][1],
                            wf[set][ni][2][0], wf[set][ni][2][1], wf[set][ni][3][0], wf[set][ni][3][1]};
          const v8i b = v8i{xf[set][mi][0][0], xf[set][mi][0][1], xf[set][mi][0][2], xf[set][mi][0][3],
                            xf[set][mi][1][0], xf[set][mi][1][1], xf[set][mi][1][2], xf[set][mi][1][3]};
          acc[mi][ni] = __builtin_amdgcn_mfma_scale_f32_16x16x128_f8f6f4(a, b, acc[mi][ni], 1, 0, 0, 127, 0, 127);
        }
    };
    auto epilogue = [&]() {
      unsigned char* stg = smem + STG_OFF;
#pragma unroll
      for (int mi = 0; mi < MI; ++mi)
#pragma unroll
        for (int ni = 0; ni < NI; ++ni) {
          f4 o;
#pragma unroll
          for (int e = 0; e < 4; ++e) {
            float y = acc[mi][ni][e] * scale;
            float yc = fminf(fmaxf(y, -60.f), 60.f);
            o[e] = y / (1.0f + __expf(-yc));
          }
          const int m = wm * WM + mi * 16 + (lane & 15);
          const int n = wn * WN + ni * 16 + 4 * (lane >> 4);
          *reinterpret_cast<f4*>(stg + m * SPITCH + n * 4) = o;
          acc[mi][ni] = f4{0.f, 0.f, 0.f, 0.f};
        }
      wait_lgkm0();
    };

    for (int step = 0; step < TOTAL; step += 2) {
#pragma unroll
      for (int u = 0; u < 2; ++u) {
        const int st = step + u;
        if (u == 0) bar();
        read_frags(u, st);
        if (st > 0) {
          mma(u ^ 1);
          if ((st % KSTEPS) == 0) {
            epilogue();
          }
        }
        tie_frags(u);
      }
    }
    mma(1);
    if (ALIAS_STG) bar();
    epilogue();
    bar();
    bar();
  } else if (wv < NWC + NWL) {
    const int lw = wv - NWC;
    constexpr int WCH = TN / 16;
    constexpr int WROWS_PP = 64 / WCH;
    auto issue = [&](int step) {
      const int r = step / KSTEPS;
      const int kt = step % KSTEPS;
      const int t = r * WG_PER_DEST + idx;
      const int tm = t / TILES_N, tn = t % TILES_N;
      unsigned char* sbase = smem + (step % NST) * STAGE;
#pragma unroll
      for (int i = 0; i < PER; ++i) {
        const int p = lw + NWL * i;
        if (p < XP) {
          const int row = 8 * p + (lane >> 3);
          const int c = (lane & 7) ^ swz_x(row);
          const unsigned char* g = P.x + (size_t)(tm * TM + row) * KDIM + kt * KSTEP + c * 16;
          __builtin_amdgcn_global_load_lds((const void*)g, (__attribute__((address_space(3))) void*)(sbase + p * 1024), 16, 0, 0);
        } else {
          const int pw = p - XP;
          const int krow = WROWS_PP * pw + lane / WCH;
          const int c = (lane % WCH) ^ swz_w<TN>(krow);
          const unsigned char* g = P.w + (size_t)(kt * KSTEP + krow) * NDIM + dest * NBLK + tn * TN + c * 16;
          __builtin_amdgcn_global_load_lds((const void*)g, (__attribute__((address_space(3))) void*)(sbase + XBYTES + pw * 1024), 16, 0, 0);
        }
      }
    };
#pragma unroll
    for (int s = 0; s < NST - 2; ++s) issue(s);
    for (int st = 0; st < TOTAL; st += 2) {
      if (st + NST - 2 < TOTAL) wait_vm<PER*(NST - 4)>(); else wait_vm<0>();
      bar();
      if (st + NST - 2 < TOTAL) { issue(st + NST - 2); issue(st + NST - 1); }
    }
    if (ALIAS_STG) bar();
    bar();
    bar();
  } else {
    float* outp = P.out_all[dest];
    const unsigned char* stg = smem + STG_OFF;
    auto push = [&](int r) {
      const int t = r * WG_PER_DEST + idx;
      const int tm = t / TILES_N, tn = t % TILES_N;
      float* tile = outp + (size_t)(rank * MLOC + tm * TM) * NBLK + tn * TN;
      constexpr int LPR = TN / 4;
      constexpr int RPI = 64 / LPR;
      constexpr int NI_ST = TM / RPI;
      f4 v[NI_ST];
#pragma unroll
      for (int i = 0; i < NI_ST; ++i) {
        const int row = i * RPI + lane / LPR, c = lane % LPR;
        v[i] = *reinterpret_cast<const f4*>(stg + row * SPITCH + c * 16);
      }
      wait_lgkm0();
#pragma unroll
      for (int i = 0; i < NI_ST; ++i) {
        const int row = i * RPI + lane / LPR, c = lane % LPR;
        store16_wt(tile + (size_t)row * NBLK + c * 4, v[i]);
      }
    };
    for (int step = 0; step < TOTAL; step += 2) {
      bar();
      if (step > 2 && (step % KSTEPS) == 2) push(step / KSTEPS - 1);
    }
    if (ALIAS_STG) bar();
    bar();
    bar();
    push(ROUNDS - 1);
    wait_vm<0>();
    if (lane == 0) {
      unsigned char* myws = P.ws_all[rank];
      unsigned int* done = reinterpret_cast<unsigned int*>(myws + WS_DONE);
      unsigned int* tmo = reinterpret_cast<unsigned int*>(myws + WS_TMO);
      unsigned int ticket = __hip_atomic_fetch_add(done, 1u, __ATOMIC_RELAXED, __HIP_MEMORY_SCOPE_AGENT);
      if ((ticket % NWG) == NWG - 1) {
        const unsigned int call = ticket / NWG + 1;
        for (int p = 0; p < NRANK; ++p) {
          if (p == rank) continue;
          unsigned int* f = reinterpret_cast<unsigned int*>(P.ws_all[p] + WS_FLAG0 + WS_FLAG_STRIDE * rank);
          __hip_atomic_store(f, call, __ATOMIC_RELAXED, __HIP_MEMORY_SCOPE_SYSTEM);
        }
        const unsigned long long t0 = __builtin_amdgcn_s_memrealtime();
        bool timed_out = false;
        for (int p = 0; p < NRANK && !timed_out; ++p) {
          if (p == rank) continue;
          unsigned int* f = reinterpret_cast<unsigned int*>(myws + WS_FLAG0 + WS_FLAG_STRIDE * p);
          while (__hip_atomic_load(f, __ATOMIC_RELAXED, __HIP_MEMORY_SCOPE_SYSTEM) < call) {
            if (__hip_atomic_load(tmo, __ATOMIC_RELAXED, __HIP_MEMORY_SCOPE_AGENT) == call ||
                __builtin_amdgcn_s_memrealtime() - t0 > 100000000ull) {
              __hip_atomic_store(tmo, call, __ATOMIC_RELAXED, __HIP_MEMORY_SCOPE_AGENT);
              timed_out = true;
              break;
            }
            __builtin_amdgcn_s_sleep(2);
          }
        }
        if (timed_out) {
          float* myout = P.out_all[rank];
          myout[(size_t)(rank * MLOC) * NBLK] = __builtin_nanf("");
        }
      }
    }
  }
}

}

extern "C" void kernel_launch(int rank, int nranks, ncclComm_t comm,
                              hipStream_t stream,
                              void* const* d_in, const int* in_sizes, int n_in,
                              void* d_out, int out_size,
                              const int* in_dtypes, int out_dtype,
                              void* const* d_out_all,
                              void* const* d_ws_all, size_t ws_bytes) {
  (void)comm; (void)in_sizes; (void)n_in; (void)d_out; (void)out_size; (void)in_dtypes; (void)out_dtype; (void)ws_bytes;
  if (nranks != NRANK) return;
  Params P;
  P.x = (const unsigned char*)d_in[0];
  P.w = (const unsigned char*)d_in[1];
  P.scale_x = (const float*)d_in[2];
  P.scale_w = (const float*)d_in[3];
  for (int p = 0; p < NRANK; ++p) {
    P.out_all[p] = (float*)d_out_all[p];
    P.ws_all[p] = (unsigned char*)d_ws_all[p];
  }
  P.rank = rank;
  constexpr int TM = TILE_M, TN = TILE_N, NST = NSTAGE;
  constexpr int ROUNDS_H = (MLOC / TM) * (NBLK / TN) / WG_PER_DEST;
  constexpr int RING = NST * (TM + TN) * 128, STG = TM * (TN * 4 + 16);
  constexpr int SMEM = ROUNDS_H == 1 ? (RING > STG ? RING : STG) : RING + STG;
  static bool attr_set[64] = {};
  int dev = 0;
  HIP_CHECK(hipGetDevice(&dev));
  if (dev >= 0 && dev < 64 && !attr_set[dev]) {
    HIP_CHECK(hipFuncSetAttribute((const void*)gemm_a2a_kernel<TM, TN, NST>, hipFuncAttributeMaxDynamicSharedMemorySize, SMEM));
    attr_set[dev] = true;
  }
  hipLaunchKernelGGL((gemm_a2a_kernel<TM, TN, NST>), dim3(NWG), dim3(NTHREADS), SMEM, stream, P);
}
```

```cpp
#include <hip/hip_runtime.h>
#include <rccl/rccl.h>
#include <cstdio>
#include <cstdint>

#define HIP_CHECK(expr) do { hipError_t _e = (expr); if (_e != hipSuccess) { \
    fprintf(stderr, "HIP error %s at %s:%d\n", hipGetErrorString(_e), __FILE__, __LINE__); \
    return; } } while (0)

namespace {

typedef int v8i __attribute__((ext_vector_type(8)));
typedef int v4i __attribute__((ext_vector_type(4)));
typedef int v2i __attribute__((ext_vector_type(2)));
typedef float f4 __attribute__((ext_vector_type(4)));
typedef __attribute__((address_space(3))) v2i* lds_v2i_ptr;

constexpr int NRANK = 8;
constexpr int MLOC = 512;
constexpr int KDIM = 4096;
constexpr int NDIM = 4096;
constexpr int NBLK = 512;
constexpr int NWG = 256;
constexpr int WG_PER_DEST = NWG / NRANK;
constexpr int KSTEP = 128;
constexpr int KSTEPS = KDIM / KSTEP;

#define TILE_M 64
#define TILE_N 128
#define NSTAGE 6
#define NWAVE_M 2
#define NWAVE_N 4
#define NWAVE_L 6
#define PF_MOD ""
constexpr int NWC = NWAVE_M * NWAVE_N;
constexpr int NWL = NWAVE_L;
constexpr int NTHREADS = (NWC + NWL + 1) * 64;

constexpr size_t WS_DONE = 0;
constexpr size_t WS_TMO = 128;
constexpr size_t WS_FLAG0 = 256;
constexpr size_t WS_FLAG_STRIDE = 128;

struct Params {
  const unsigned char* x;
  const unsigned char* w;
  const float* scale_x;
  const float* scale_w;
  float* out_all[NRANK];
  unsigned char* ws_all[NRANK];
  int rank;
};

__device__ __forceinline__ int swz_x(int row) { return ((row >> 1) & 1) | (row & 4); }
template <int TN>
__device__ __forceinline__ int swz_w(int k) {
  if (TN == 64) return ((k >> 2) & 1) | (((k >> 5) & 1) << 1);
  if (TN == 128) return ((k >> 1) & 1) | (((k >> 2) & 1) << 1) | (((k >> 5) & 1) << 2);
  return ((k >> 5) & 1);
}

template <int N>
__device__ __forceinline__ void wait_vm() { asm volatile("s_waitcnt vmcnt(%0)" :: "n"(N) : "memory"); }
__device__ __forceinline__ void wait_lgkm0() { asm volatile("s_waitcnt lgkmcnt(0)" ::: "memory"); }

__device__ __forceinline__ void bar() { asm volatile("s_barrier" ::: "memory"); }

__device__ __forceinline__ void store16_wt(float* p, f4 v) {
  asm volatile("global_store_dwordx4 %0, %1, off sc0 sc1\n s_nop 1" :: "v"(p), "v"(v) : "memory");
}

template <int TM, int TN, int NST>
__global__ void __launch_bounds__(NTHREADS, 1) gemm_a2a_kernel(Params P) {
  constexpr int TILES_N = NBLK / TN;
  constexpr int TILES_PER_DEST = (MLOC / TM) * TILES_N;
  constexpr int ROUNDS = TILES_PER_DEST / WG_PER_DEST;
  constexpr int TOTAL = ROUNDS * KSTEPS;
  constexpr int XBYTES = TM * KSTEP;
  constexpr int WBYTES = KSTEP * TN;
  constexpr int STAGE = XBYTES + WBYTES;
  constexpr int XP = TM / 8;
  constexpr int WP = TN / 8;
  constexpr int PER = (XP + WP) / NWL;
  constexpr int WM = TM / NWAVE_M, WN = TN / NWAVE_N;
  constexpr int MI = WM / 16, NI = WN / 16;
  constexpr int SPITCH = TN * 4 + 16;
  constexpr bool ALIAS_STG = (ROUNDS == 1);
  constexpr int STG_OFF = ALIAS_STG ? 0 : NST * STAGE;
  static_assert((XP + WP) % NWL == 0, "pieces must split over the loader waves");
  static_assert(PER * (NST - 1) < 60, "vmcnt range");

  extern __shared__ __attribute__((aligned(1024))) unsigned char smem[];

  const int tid = threadIdx.x;
  const int lane = tid & 63;
  const int wv = tid >> 6;
  const int wg = blockIdx.x;
  const int dest = wg % NRANK;
  const int idx = wg / NRANK;
  const int rank = P.rank;

  if (wv < NWC) {
    const int wm = wv / NWAVE_N, wn = wv % NWAVE_N;
    const float scale = P.scale_x[0] * P.scale_w[0];
    f4 acc[MI][NI];
#pragma unroll
    for (int a = 0; a < MI; ++a)
#pragma unroll
      for (int b = 0; b < NI; ++b) acc[a][b] = f4{0.f, 0.f, 0.f, 0.f};

    unsigned xoff[MI][2];
#pragma unroll
    for (int mi = 0; mi < MI; ++mi)
#pragma unroll
      for (int h = 0; h < 2; ++h) {
        const int row = wm * WM + mi * 16 + (lane & 15);
        const int c = 2 * (lane >> 4) + h;
        xoff[mi][h] = row * KSTEP + ((c ^ swz_x(row)) * 16);
      }
    unsigned woff[NI][4];
#pragma unroll
    for (int ni = 0; ni < NI; ++ni)
#pragma unroll
      for (int q = 0; q < 4; ++q) {
        const int g = lane >> 4, j = lane & 15;
        const int krow = 32 * g + 8 * q + (j >> 1);
        const int nt = (wn * WN + ni * 16) / 16;
        woff[ni][q] = XBYTES + krow * TN + ((nt ^ swz_w<TN>(krow)) * 16) + 8 * (j & 1);
      }

    v4i xf[2][MI][2];
    v2i wf[2][NI][4];

    auto read_frags = [&](int set, int step) {
      const unsigned sb = (unsigned)((step % NST) * STAGE);
#pragma unroll
      for (int ni = 0; ni < NI; ++ni)
#pragma unroll
        for (int q = 0; q < 4; ++q)
          asm volatile("ds_read_b64_tr_b8 %0, %1" : "=v"(wf[set][ni][q]) : "v"(woff[ni][q] + sb) : "memory");
#pragma unroll
      for (int mi = 0; mi < MI; ++mi)
#pragma unroll
        for (int h = 0; h < 2; ++h)
          asm volatile("ds_read_b128 %0, %1" : "=v"(xf[set][mi][h]) : "v"(xoff[mi][h] + sb) : "memory");
    };
    auto tie_frags = [&](int set) {
      asm volatile("s_waitcnt lgkmcnt(0)" ::: "memory");
#pragma unroll
      for (int ni = 0; ni < NI; ++ni)
#pragma unroll
        for (int q = 0; q < 4; ++q) asm volatile("" : "+v"(wf[set][ni][q]));
#pragma unroll
      for (int mi = 0; mi < MI; ++mi)
#pragma unroll
        for (int h = 0; h < 2; ++h) asm volatile("" : "+v"(xf[set][mi][h]));
    };
    auto mma = [&](int set) {
#pragma unroll
      for (int mi = 0; mi < MI; ++mi)
#pragma unroll
        for (int ni = 0; ni < NI; ++ni) {
          const v8i a = v8i{wf[set][ni][0][0], wf[set][ni][0][1], wf[set][ni][1][0], wf[set][ni][1][1],
                            wf[set][ni][2][0], wf[set][ni][2][1], wf[set][ni][3][0], wf[set][ni][3][1]};
          const v8i b = v8i{xf[set][mi][0][0], xf[set][mi][0][1], xf[set][mi][0][2], xf[set][mi][0][3],
                            xf[set][mi][1][0], xf[set][mi][1][1], xf[set][mi][1][2], xf[set][mi][1][3]};
          acc[mi][ni] = __builtin_amdgcn_mfma_scale_f32_16x16x128_f8f6f4(a, b, acc[mi][ni], 1, 0, 0, 127, 0, 127);
        }
    };
    auto epilogue = [&]() {
      unsigned char* stg = smem + STG_OFF;
#pragma unroll
      for (int mi = 0; mi < MI; ++mi)
#pragma unroll
        for (int ni = 0; ni < NI; ++ni) {
          f4 o;
#pragma unroll
          for (int e = 0; e < 4; ++e) {
            float y = acc[mi][ni][e] * scale;
            float yc = fminf(fmaxf(y, -60.f), 60.f);
            o[e] = y / (1.0f + __expf(-yc));
          }
          const int m = wm * WM + mi * 16 + (lane & 15);
          const int n = wn * WN + ni * 16 + 4 * (lane >> 4);
          *reinterpret_cast<f4*>(stg + m * SPITCH + n * 4) = o;
          acc[mi][ni] = f4{0.f, 0.f, 0.f, 0.f};
        }
      wait_lgkm0();
    };

    for (int step = 0; step < TOTAL; step += 2) {
#pragma unroll
      for (int u = 0; u < 2; ++u) {
        const int st = step + u;
        if (u == 0) bar();
        read_frags(u, st);
        if (st > 0) {
          mma(u ^ 1);
          if ((st % KSTEPS) == 0) {
            epilogue();
          }
        }
        tie_frags(u);
      }
    }
    mma(1);
    if (ALIAS_STG) bar();
    epilogue();
    bar();
    bar();
  } else if (wv < NWC + NWL) {
    const int lw = wv - NWC;
    constexpr int WCH = TN / 16;
    constexpr int WROWS_PP = 64 / WCH;
    auto issue = [&](int step) {
      const int r = step / KSTEPS;
      const int kt = step % KSTEPS;
      const int t = r * WG_PER_DEST + idx;
      const int tm = t / TILES_N, tn = t % TILES_N;
      unsigned char* sbase = smem + (step % NST) * STAGE;
#pragma unroll
      for (int i = 0; i < PER; ++i) {
        const int p = lw + NWL * i;
        if (p < XP) {
          const int row = 8 * p + (lane >> 3);
          const int c = (lane & 7) ^ swz_x(row);
          const unsigned char* g = P.x + (size_t)(tm * TM + row) * KDIM + kt * KSTEP + c * 16;
          __builtin_amdgcn_global_load_lds((const void*)g, (__attribute__((address_space(3))) void*)(sbase + p * 1024), 16, 0, 0);
        } else {
          const int pw = p - XP;
          const int krow = WROWS_PP * pw + lane / WCH;
          const int c = (lane % WCH) ^ swz_w<TN>(krow);
          const unsigned char* g = P.w + (size_t)(kt * KSTEP + krow) * NDIM + dest * NBLK + tn * TN + c * 16;
          __builtin_amdgcn_global_load_lds((const void*)g, (__attribute__((address_space(3))) void*)(sbase + XBYTES + pw * 1024), 16, 0, 0);
        }
      }
    };
#pragma unroll
    for (int s = 0; s < NST - 2; ++s) issue(s);
    for (int st = 0; st < TOTAL; st += 2) {
      if (st + NST - 2 < TOTAL) wait_vm<PER*(NST - 4)>(); else wait_vm<0>();
      bar();
      if (st + NST - 2 < TOTAL) { issue(st + NST - 2); issue(st + NST - 1); }
    }
    if (ALIAS_STG) bar();
    bar();
    bar();
  } else {
    float* outp = P.out_all[dest];
    const unsigned char* stg = smem + STG_OFF;
    v4i junk = v4i{0, 0, 0, 0};
    {
      const unsigned char* pw = P.w + (size_t)wg * 65536 + lane * 16;
#pragma unroll
      for (int j = 0; j < 64; ++j)
        asm volatile("global_load_dwordx4 %0, %1, off offset:0 " PF_MOD : "+v"(junk) : "v"(pw + j * 1024) : "memory");
      const unsigned char* px = P.x + (size_t)wg * 8192 + lane * 16;
#pragma unroll
      for (int j = 0; j < 8; ++j)
        asm volatile("global_load_dwordx4 %0, %1, off offset:0 " PF_MOD : "+v"(junk) : "v"(px + j * 1024) : "memory");
    }
    auto push = [&](int r) {
      const int t = r * WG_PER_DEST + idx;
      const int tm = t / TILES_N, tn = t % TILES_N;
      float* tile = outp + (size_t)(rank * MLOC + tm * TM) * NBLK + tn * TN;
      constexpr int LPR = TN / 4;
      constexpr int RPI = 64 / LPR;
      constexpr int NI_ST = TM / RPI;
      f4 v[NI_ST];
#pragma unroll
      for (int i = 0; i < NI_ST; ++i) {
        const int row = i * RPI + lane / LPR, c = lane % LPR;
        v[i] = *reinterpret_cast<const f4*>(stg + row * SPITCH + c * 16);
      }
      wait_lgkm0();
#pragma unroll
      for (int i = 0; i < NI_ST; ++i) {
        const int row = i * RPI + lane / LPR, c = lane % LPR;
        store16_wt(tile + (size_t)row * NBLK + c * 4, v[i]);
      }
    };
    for (int step = 0; step < TOTAL; step += 2) {
      bar();
      if (step > 2 && (step % KSTEPS) == 2) push(step / KSTEPS - 1);
    }
    if (ALIAS_STG) bar();
    bar();
    bar();
    asm volatile("s_waitcnt vmcnt(0)" : "+v"(junk) :: "memory");
    push(ROUNDS - 1);
    wait_vm<0>();
    if (lane == 0) {
      unsigned char* myws = P.ws_all[rank];
      unsigned int* done = reinterpret_cast<unsigned int*>(myws + WS_DONE);
      unsigned int* tmo = reinterpret_cast<unsigned int*>(myws + WS_TMO);
      unsigned int ticket = __hip_atomic_fetch_add(done, 1u, __ATOMIC_RELAXED, __HIP_MEMORY_SCOPE_AGENT);
      if ((ticket % NWG) == NWG - 1) {
        const unsigned int call = ticket / NWG + 1;
        for (int p = 0; p < NRANK; ++p) {
          if (p == rank) continue;
          unsigned int* f = reinterpret_cast<unsigned int*>(P.ws_all[p] + WS_FLAG0 + WS_FLAG_STRIDE * rank);
          __hip_atomic_store(f, call, __ATOMIC_RELAXED, __HIP_MEMORY_SCOPE_SYSTEM);
        }
        const unsigned long long t0 = __builtin_amdgcn_s_memrealtime();
        bool timed_out = false;
        for (int p = 0; p < NRANK && !timed_out; ++p) {
          if (p == rank) continue;
          unsigned int* f = reinterpret_cast<unsigned int*>(myws + WS_FLAG0 + WS_FLAG_STRIDE * p);
          while (__hip_atomic_load(f, __ATOMIC_RELAXED, __HIP_MEMORY_SCOPE_SYSTEM) < call) {
            if (__hip_atomic_load(tmo, __ATOMIC_RELAXED, __HIP_MEMORY_SCOPE_AGENT) == call ||
                __builtin_amdgcn_s_memrealtime() - t0 > 100000000ull) {
              __hip_atomic_store(tmo, call, __ATOMIC_RELAXED, __HIP_MEMORY_SCOPE_AGENT);
              timed_out = true;
              break;
            }
            __builtin_amdgcn_s_sleep(2);
          }
        }
        if (timed_out) {
          float* myout = P.out_all[rank];
          myout[(size_t)(rank * MLOC) * NBLK] = __builtin_nanf("");
        }
      }
    }
  }
}

}

extern "C" void kernel_launch(int rank, int nranks, ncclComm_t comm,
                              hipStream_t stream,
                              void* const* d_in, const int* in_sizes, int n_in,
                              void* d_out, int out_size,
                              const int* in_dtypes, int out_dtype,
                              void* const* d_out_all,
                              void* const* d_ws_all, size_t ws_bytes) {
  (void)comm; (void)in_sizes; (void)n_in; (void)d_out; (void)out_size; (void)in_dtypes; (void)out_dtype; (void)ws_bytes;
  if (nranks != NRANK) return;
  Params P;
  P.x = (const unsigned char*)d_in[0];
  P.w = (const unsigned char*)d_in[1];
  P.scale_x = (const float*)d_in[2];
  P.scale_w = (const float*)d_in[3];
  for (int p = 0; p < NRANK; ++p) {
    P.out_all[p] = (float*)d_out_all[p];
    P.ws_all[p] = (unsigned char*)d_ws_all[p];
  }
  P.rank = rank;
  constexpr int TM = TILE_M, TN = TILE_N, NST = NSTAGE;
  constexpr int ROUNDS_H = (MLOC / TM) * (NBLK / TN) / WG_PER_DEST;
  constexpr int RING = NST * (TM + TN) * 128, STG = TM * (TN * 4 + 16);
  constexpr int SMEM = ROUNDS_H == 1 ? (RING > STG ? RING : STG) : RING + STG;
  static bool attr_set[64] = {};
  int dev = 0;
  HIP_CHECK(hipGetDevice(&dev));
  if (dev >= 0 && dev < 64 && !attr_set[dev]) {
    HIP_CHECK(hipFuncSetAttribute((const void*)gemm_a2a_kernel<TM, TN, NST>, hipFuncAttributeMaxDynamicSharedMemorySize, SMEM));
    attr_set[dev] = true;
  }
  hipLaunchKernelGGL((gemm_a2a_kernel<TM, TN, NST>), dim3(NWG), dim3(NTHREADS), SMEM, stream, P);
}
```

```cpp
#include <hip/hip_runtime.h>
#include <rccl/rccl.h>
#include <cstdio>
#include <cstdint>

#define HIP_CHECK(expr) do { hipError_t _e = (expr); if (_e != hipSuccess) { \
    fprintf(stderr, "HIP error %s at %s:%d\n", hipGetErrorString(_e), __FILE__, __LINE__); \
    return; } } while (0)

namespace {

typedef int v8i __attribute__((ext_vector_type(8)));
typedef int v4i __attribute__((ext_vector_type(4)));
typedef int v2i __attribute__((ext_vector_type(2)));
typedef float f4 __attribute__((ext_vector_type(4)));
typedef __attribute__((address_space(3))) v2i* lds_v2i_ptr;

constexpr int NRANK = 8;
constexpr int MLOC = 512;
constexpr int KDIM = 4096;
constexpr int NDIM = 4096;
constexpr int NBLK = 512;
constexpr int NWG = 256;
constexpr int WG_PER_DEST = NWG / NRANK;
constexpr int KSTEP = 128;
constexpr int KSTEPS = KDIM / KSTEP;

#define TILE_M 64
#define TILE_N 128
#define NSTAGE 6
#define NWAVE_M 2
#define NWAVE_N 4
#define NWAVE_L 6
constexpr int NWC = NWAVE_M * NWAVE_N;
constexpr int NWL = NWAVE_L;
constexpr int NTHREADS = (NWC + NWL + 1) * 64;

constexpr size_t WS_CNT = 0;
constexpr size_t WS_TMO = 128;
constexpr size_t WS_CALL = 256;

struct Params {
  const unsigned char* x;
  const unsigned char* w;
  const float* scale_x;
  const float* scale_w;
  float* out_all[NRANK];
  unsigned char* ws_all[NRANK];
  int rank;
};

__device__ __forceinline__ int swz_x(int row) { return ((row >> 1) & 1) | (row & 4); }
template <int TN>
__device__ __forceinline__ int swz_w(int k) {
  if (TN == 64) return ((k >> 2) & 1) | (((k >> 5) & 1) << 1);
  if (TN == 128) return ((k >> 1) & 1) | (((k >> 2) & 1) << 1) | (((k >> 5) & 1) << 2);
  return ((k >> 5) & 1);
}

template <int N>
__device__ __forceinline__ void wait_vm() { asm volatile("s_waitcnt vmcnt(%0)" :: "n"(N) : "memory"); }
__device__ __forceinline__ void wait_lgkm0() { asm volatile("s_waitcnt lgkmcnt(0)" ::: "memory"); }

__device__ __forceinline__ void bar() { asm volatile("s_barrier" ::: "memory"); }

__device__ __forceinline__ void store16_wt(float* p, f4 v) {
  asm volatile("global_store_dwordx4 %0, %1, off sc0 sc1\n s_nop 1" :: "v"(p), "v"(v) : "memory");
}

template <int TM, int TN, int NST>
__global__ void __launch_bounds__(NTHREADS, 1) gemm_a2a_kernel(Params P) {
  constexpr int TILES_N = NBLK / TN;
  constexpr int TILES_PER_DEST = (MLOC / TM) * TILES_N;
  constexpr int ROUNDS = TILES_PER_DEST / WG_PER_DEST;
  constexpr int TOTAL = ROUNDS * KSTEPS;
  constexpr int XBYTES = TM * KSTEP;
  constexpr int WBYTES = KSTEP * TN;
  constexpr int STAGE = XBYTES + WBYTES;
  constexpr int XP = TM / 8;
  constexpr int WP = TN / 8;
  constexpr int PER = (XP + WP) / NWL;
  constexpr int WM = TM / NWAVE_M, WN = TN / NWAVE_N;
  constexpr int MI = WM / 16, NI = WN / 16;
  constexpr int SPITCH = TN * 4 + 16;
  constexpr bool ALIAS_STG = (ROUNDS == 1);
  constexpr int STG_OFF = ALIAS_STG ? 0 : NST * STAGE;
  static_assert((XP + WP) % NWL == 0, "pieces must split over the loader waves");
  static_assert(PER * (NST - 1) < 60, "vmcnt range");

  extern __shared__ __attribute__((aligned(1024))) unsigned char smem[];

  const int tid = threadIdx.x;
  const int lane = tid & 63;
  const int wv = tid >> 6;
  const int wg = blockIdx.x;
  const int dest = wg % NRANK;
  const int idx = wg / NRANK;
  const int rank = P.rank;

  if (wv < NWC) {
    const int wm = wv / NWAVE_N, wn = wv % NWAVE_N;
    const float scale = P.scale_x[0] * P.scale_w[0];
    f4 acc[MI][NI];
#pragma unroll
    for (int a = 0; a < MI; ++a)
#pragma unroll
      for (int b = 0; b < NI; ++b) acc[a][b] = f4{0.f, 0.f, 0.f, 0.f};

    unsigned xoff[MI][2];
#pragma unroll
    for (int mi = 0; mi < MI; ++mi)
#pragma unroll
      for (int h = 0; h < 2; ++h) {
        const int row = wm * WM + mi * 16 + (lane & 15);
        const int c = 2 * (lane >> 4) + h;
        xoff[mi][h] = row * KSTEP + ((c ^ swz_x(row)) * 16);
      }
    unsigned woff[NI][4];
#pragma unroll
    for (int ni = 0; ni < NI; ++ni)
#pragma unroll
      for (int q = 0; q < 4; ++q) {
        const int g = lane >> 4, j = lane & 15;
        const int krow = 32 * g + 8 * q + (j >> 1);
        const int nt = (wn * WN + ni * 16) / 16;
        woff[ni][q] = XBYTES + krow * TN + ((nt ^ swz_w<TN>(krow)) * 16) + 8 * (j & 1);
      }

    v4i xf[2][MI][2];
    v2i wf[2][NI][4];

    auto read_frags = [&](int set, int step) {
      const unsigned sb = (unsigned)((step % NST) * STAGE);
#pragma unroll
      for (int ni = 0; ni < NI; ++ni)
#pragma unroll
        for (int q = 0; q < 4; ++q)
          asm volatile("ds_read_b64_tr_b8 %0, %1" : "=v"(wf[set][ni][q]) : "v"(woff[ni][q] + sb) : "memory");
#pragma unroll
      for (int mi = 0; mi < MI; ++mi)
#pragma unroll
        for (int h = 0; h < 2; ++h)
          asm volatile("ds_read_b128 %0, %1" : "=v"(xf[set][mi][h]) : "v"(xoff[mi][h] + sb) : "memory");
    };
    auto tie_frags = [&](int set) {
      asm volatile("s_waitcnt lgkmcnt(0)" ::: "memory");
#pragma unroll
      for (int ni = 0; ni < NI; ++ni)
#pragma unroll
        for (int q = 0; q < 4; ++q) asm volatile("" : "+v"(wf[set][ni][q]));
#pragma unroll
      for (int mi = 0; mi < MI; ++mi)
#pragma unroll
        for (int h = 0; h < 2; ++h) asm volatile("" : "+v"(xf[set][mi][h]));
    };
    auto mma = [&](int set) {
#pragma unroll
      for (int mi = 0; mi < MI; ++mi)
#pragma unroll
        for (int ni = 0; ni < NI; ++ni) {
          const v8i a = v8i{wf[set][ni][0][0], wf[set][ni][0][1], wf[set][ni][1][0], wf[set][ni][1][1],
                            wf[set][ni][2][0], wf[set][ni][2][1], wf[set][ni][3][0], wf[set][ni][3][1]};
          const v8i b = v8i{xf[set][mi][0][0], xf[set][mi][0][1], xf[set][mi][0][2], xf[set][mi][0][3],
                            xf[set][mi][1][0], xf[set][mi][1][1], xf[set][mi][1][2], xf[set][mi][1][3]};
          acc[mi][ni] = __builtin_amdgcn_mfma_scale_f32_16x16x128_f8f6f4(a, b, acc[mi][ni], 1, 0, 0, 127, 0, 127);
        }
    };
    auto epilogue = [&]() {
      unsigned char* stg = smem + STG_OFF;
#pragma unroll
      for (int mi = 0; mi < MI; ++mi)
#pragma unroll
        for (int ni = 0; ni < NI; ++ni) {
          f4 o;
#pragma unroll
          for (int e = 0; e < 4; ++e) {
            float y = acc[mi][ni][e] * scale;
            float yc = fminf(fmaxf(y, -60.f), 60.f);
            o[e] = y / (1.0f + __expf(-yc));
          }
          const int m = wm * WM + mi * 16 + (lane & 15);
          const int n = wn * WN + ni * 16 + 4 * (lane >> 4);
          *reinterpret_cast<f4*>(stg + m * SPITCH + n * 4) = o;
          acc[mi][ni] = f4{0.f, 0.f, 0.f, 0.f};
        }
      wait_lgkm0();
    };

    for (int step = 0; step < TOTAL; step += 2) {
#pragma unroll
      for (int u = 0; u < 2; ++u) {
        const int st = step + u;
        if (u == 0) bar();
        read_frags(u, st);
        if (st > 0) {
          mma(u ^ 1);
          if ((st % KSTEPS) == 0) {
            epilogue();
          }
        }
        tie_frags(u);
      }
    }
    mma(1);
    if (ALIAS_STG) bar();
    epilogue();
    bar();
    bar();
  } else if (wv < NWC + NWL) {
    const int lw = wv - NWC;
    constexpr int WCH = TN / 16;
    constexpr int WROWS_PP = 64 / WCH;
    auto issue = [&](int step) {
      const int r = step / KSTEPS;
      const int kt = step % KSTEPS;
      const int t = r * WG_PER_DEST + idx;
      const int tm = t / TILES_N, tn = t % TILES_N;
      unsigned char* sbase = smem + (step % NST) * STAGE;
#pragma unroll
      for (int i = 0; i < PER; ++i) {
        const int p = lw + NWL * i;
        if (p < XP) {
          const int row = 8 * p + (lane >> 3);
          const int c = (lane & 7) ^ swz_x(row);
          const unsigned char* g = P.x + (size_t)(tm * TM + row) * KDIM + kt * KSTEP + c * 16;
          __builtin_amdgcn_global_load_lds((const void*)g, (__attribute__((address_space(3))) void*)(sbase + p * 1024), 16, 0, 0);
        } else {
          const int pw = p - XP;
          const int krow = WROWS_PP * pw + lane / WCH;
          const int c = (lane % WCH) ^ swz_w<TN>(krow);
          const unsigned char* g = P.w + (size_t)(kt * KSTEP + krow) * NDIM + dest * NBLK + tn * TN + c * 16;
          __builtin_amdgcn_global_load_lds((const void*)g, (__attribute__((address_space(3))) void*)(sbase + XBYTES + pw * 1024), 16, 0, 0);
        }
      }
    };
#pragma unroll
    for (int s = 0; s < NST - 2; ++s) issue(s);
    for (int st = 0; st < TOTAL; st += 2) {
      if (st + NST - 2 < TOTAL) wait_vm<PER*(NST - 4)>(); else wait_vm<0>();
      bar();
      if (st + NST - 2 < TOTAL) { issue(st + NST - 2); issue(st + NST - 1); }
    }
    if (ALIAS_STG) bar();
    bar();
    bar();
  } else {
    float* outp = P.out_all[dest];
    const unsigned char* stg = smem + STG_OFF;
    auto push = [&](int r) {
      const int t = r * WG_PER_DEST + idx;
      const int tm = t / TILES_N, tn = t % TILES_N;
      float* tile = outp + (size_t)(rank * MLOC + tm * TM) * NBLK + tn * TN;
      constexpr int LPR = TN / 4;
      constexpr int RPI = 64 / LPR;
      constexpr int NI_ST = TM / RPI;
      f4 v[NI_ST];
#pragma unroll
      for (int i = 0; i < NI_ST; ++i) {
        const int row = i * RPI + lane / LPR, c = lane % LPR;
        v[i] = *reinterpret_cast<const f4*>(stg + row * SPITCH + c * 16);
      }
      wait_lgkm0();
#pragma unroll
      for (int i = 0; i < NI_ST; ++i) {
        const int row = i * RPI + lane / LPR, c = lane % LPR;
        store16_wt(tile + (size_t)row * NBLK + c * 4, v[i]);
      }
    };
    for (int step = 0; step < TOTAL; step += 2) {
      bar();
      if (step > 2 && (step % KSTEPS) == 2) push(step / KSTEPS - 1);
    }
    if (ALIAS_STG) bar();
    bar();
    bar();
    push(ROUNDS - 1);
    wait_vm<0>();
    if (lane == 0) {
      unsigned char* myws = P.ws_all[rank];
      if (dest != rank)
        __hip_atomic_fetch_add(reinterpret_cast<unsigned int*>(P.ws_all[dest] + WS_CNT), 1u, __ATOMIC_RELAXED, __HIP_MEMORY_SCOPE_SYSTEM);
      if (dest == rank && idx == 0) {
        unsigned int* callw = reinterpret_cast<unsigned int*>(myws + WS_CALL);
        unsigned int* cnt = reinterpret_cast<unsigned int*>(myws + WS_CNT);
        unsigned int* tmo = reinterpret_cast<unsigned int*>(myws + WS_TMO);
        const unsigned int call = __hip_atomic_load(callw, __ATOMIC_RELAXED, __HIP_MEMORY_SCOPE_AGENT) + 1;
        __hip_atomic_store(callw, call, __ATOMIC_RELAXED, __HIP_MEMORY_SCOPE_AGENT);
        const unsigned int want = call * (unsigned)((NRANK - 1) * WG_PER_DEST);
        const unsigned long long t0 = __builtin_amdgcn_s_memrealtime();
        while (__hip_atomic_load(cnt, __ATOMIC_RELAXED, __HIP_MEMORY_SCOPE_SYSTEM) < want) {
          if (__builtin_amdgcn_s_memrealtime() - t0 > 100000000ull) {
            __hip_atomic_store(tmo, call, __ATOMIC_RELAXED, __HIP_MEMORY_SCOPE_AGENT);
            P.out_all[rank][(size_t)(rank * MLOC) * NBLK] = __builtin_nanf("");
            break;
          }
          __builtin_amdgcn_s_sleep(1);
        }
      }
    }
  }
}

}

extern "C" void kernel_launch(int rank, int nranks, ncclComm_t comm,
                              hipStream_t stream,
                              void* const* d_in, const int* in_sizes, int n_in,
                              void* d_out, int out_size,
                              const int* in_dtypes, int out_dtype,
                              void* const* d_out_all,
                              void* const* d_ws_all, size_t ws_bytes) {
  (void)comm; (void)in_sizes; (void)n_in; (void)d_out; (void)out_size; (void)in_dtypes; (void)out_dtype; (void)ws_bytes;
  if (nranks != NRANK) return;
  Params P;
  P.x = (const unsigned char*)d_in[0];
  P.w = (const unsigned char*)d_in[1];
  P.scale_x = (const float*)d_in[2];
  P.scale_w = (const float*)d_in[3];
  for (int p = 0; p < NRANK; ++p) {
    P.out_all[p] = (float*)d_out_all[p];
    P.ws_all[p] = (unsigned char*)d_ws_all[p];
  }
  P.rank = rank;
  constexpr int TM = TILE_M, TN = TILE_N, NST = NSTAGE;
  constexpr int ROUNDS_H = (MLOC / TM) * (NBLK / TN) / WG_PER_DEST;
  constexpr int RING = NST * (TM + TN) * 128, STG = TM * (TN * 4 + 16);
  constexpr int SMEM = ROUNDS_H == 1 ? (RING > STG ? RING : STG) : RING + STG;
  static bool attr_set[64] = {};
  int dev = 0;
  HIP_CHECK(hipGetDevice(&dev));
  if (dev >= 0 && dev < 64 && !attr_set[dev]) {
    HIP_CHECK(hipFuncSetAttribute((const void*)gemm_a2a_kernel<TM, TN, NST>, hipFuncAttributeMaxDynamicSharedMemorySize, SMEM));
    attr_set[dev] = true;
  }
  hipLaunchKernelGGL((gemm_a2a_kernel<TM, TN, NST>), dim3(NWG), dim3(NTHREADS), SMEM, stream, P);
}
```
